```python
import numpy as np

kernel_name = "gpu_a2a_gemm_m2048_k2048_n2048_f32_silu"


ABSMAX_RTOL = 0.001

COMPUTE_DTYPE = "float32"
EPILOGUE = "silu"
IN_QUANT = None
OUT_QUANT = None
IN_DTYPES = {"x": "float32", "w_mat": "float32"}
WORLD_SIZE = 8

def setup_inputs():

    x = np.random.randn(2048, 2048).astype(np.float32)
    w_mat = (np.random.randn(2048, 2048) / np.sqrt(2048)).astype(np.float32)
    return {"x": x, "w_mat": w_mat}

def _epilogue(y):
    return (y / (1.0 + np.exp(-np.clip(y, -60.0, 60.0)))).astype(np.float32)

def reference(x, w_mat):
    y = (x @ w_mat).astype(np.float32)
    return _epilogue(y)
```

```cpp
#include <hip/hip_runtime.h>
#include <rccl/rccl.h>
#include <cstdio>
#include <cstdint>

#define HIP_CHECK(expr) do { hipError_t _e = (expr); if (_e != hipSuccess) { \
    fprintf(stderr, "HIP error %s at %s:%d\n", hipGetErrorString(_e), __FILE__, __LINE__); \
    return; } } while (0)

typedef _Float16 half8 __attribute__((ext_vector_type(8)));
typedef float f32x16 __attribute__((ext_vector_type(16)));
typedef float f32x4 __attribute__((ext_vector_type(4)));
typedef unsigned u32x4 __attribute__((ext_vector_type(4)));

namespace {

constexpr int NRANKS = 8;
constexpr int MLOC = 256;
constexpr int KBLK = 256;
constexpr int NDIM = 2048;
constexpr int NWG = 256;
constexpr int NTHREADS = 512;

constexpr unsigned A_SRC_BYTES = 32u * 256u * 16u;
constexpr unsigned A_PAR_BYTES = NRANKS * A_SRC_BYTES;
constexpr unsigned A_OFF = 0;
constexpr unsigned FLAG_OFF = 2u * A_PAR_BYTES;
constexpr unsigned FLAG_SRC_BYTES = 32u * 16u;
constexpr unsigned FLAG_PAR_BYTES = NRANKS * FLAG_SRC_BYTES;
constexpr unsigned CTR_OFF = FLAG_OFF + 2u * FLAG_PAR_BYTES;
constexpr unsigned TMO_OFF = CTR_OFF + NWG * 64u;
constexpr unsigned WS_USED = TMO_OFF + 128u;

constexpr unsigned long long DEADLINE_TICKS = 50000000ull;

constexpr int AUX_SYS = 17;

constexpr int LDS_WAVE_BYTES = 16384;
constexpr int RED_STRIDE = 36;
constexpr int LDS_MISC_OFF = 8 * LDS_WAVE_BYTES;
constexpr int LDS_BYTES = LDS_MISC_OFF + 256;

struct Params {
  const float* x;
  const float* w;
  float* out;
  char* ws[NRANKS];
  int rank;
};

__device__ __forceinline__ float wave_max(float v) {
#pragma unroll
  for (int o = 32; o >= 1; o >>= 1) v = fmaxf(v, __shfl_xor(v, o, 64));
  return v;
}

__device__ __forceinline__ float absmax4(const f32x4 v) {
  return fmaxf(fmaxf(fabsf(v[0]), fabsf(v[1])), fmaxf(fabsf(v[2]), fabsf(v[3])));
}

__global__ void __launch_bounds__(NTHREADS, 2) a2a_gemm_silu(Params p) {
  __shared__ __attribute__((aligned(16))) char lds[LDS_BYTES];

  const int tid = threadIdx.x;
  const int lane = tid & 63;
  const int wave = __builtin_amdgcn_readfirstlane(tid >> 6);
  const int b = blockIdx.x;
  const int rank = p.rank;
  char* const myws = p.ws[rank];

  float* const s_rowmax = reinterpret_cast<float*>(lds + LDS_MISC_OFF);
  unsigned* const s_call = reinterpret_cast<unsigned*>(lds + LDS_MISC_OFF + 128);
  unsigned* const s_cnt = reinterpret_cast<unsigned*>(lds + LDS_MISC_OFF + 132);

  const int xcd = b & 7, idx = b >> 3;
  const int n0 = xcd * 256 + (idx & 7) * 32;
  const int m0 = (idx >> 3) * 64;

  f32x4 xv0 = {0.f, 0.f, 0.f, 0.f}, xv1 = {0.f, 0.f, 0.f, 0.f};
  if (wave < 4) {
    const float* src = p.x + (size_t)(8 * b + (lane >> 3)) * KBLK + 64 * wave + 8 * (lane & 7);
    xv0 = *reinterpret_cast<const f32x4*>(src);
    xv1 = *reinterpret_cast<const f32x4*>(src + 4);
  }
  if (tid == 0) {
    unsigned* ctr = reinterpret_cast<unsigned*>(myws + CTR_OFF + 64u * (unsigned)b);
    const unsigned t = __hip_atomic_fetch_add(ctr, 1u, __ATOMIC_RELAXED, __HIP_MEMORY_SCOPE_AGENT);
    *s_call = t + 1u;
    *s_cnt = 0u;
  }

  f32x4 wv[4][8];
  {
    const float* wsrc = p.w + (size_t)(KBLK * wave + 8 * (lane >> 3)) * NDIM + n0 + 4 * (lane & 7);
#pragma unroll
    for (int bt = 0; bt < 4; ++bt)
#pragma unroll
      for (int j = 0; j < 8; ++j)
        wv[bt][j] = *reinterpret_cast<const f32x4*>(wsrc + (size_t)(64 * bt + j) * NDIM);
  }

  if (wave < 4) {
    float m = fmaxf(absmax4(xv0), absmax4(xv1));
    m = fmaxf(m, __shfl_xor(m, 1, 64));
    m = fmaxf(m, __shfl_xor(m, 2, 64));
    m = fmaxf(m, __shfl_xor(m, 4, 64));
    if ((lane & 7) == 0) s_rowmax[wave * 8 + (lane >> 3)] = m;
  }
  __syncthreads();
  const unsigned call = *s_call;
  const unsigned par = call & 1u;

  if (wave < 4) {
    const int r = lane >> 3;
    const float m = fmaxf(fmaxf(s_rowmax[r], s_rowmax[8 + r]), fmaxf(s_rowmax[16 + r], s_rowmax[24 + r]));
    int e = __builtin_amdgcn_frexp_expf(m);
    e = max(e, -127);
    const int sh = 15 - e;
    half8 h;
#pragma unroll
    for (int i = 0; i < 4; ++i) {
      h[i] = (_Float16)__builtin_amdgcn_ldexpf(xv0[i], sh);
      h[4 + i] = (_Float16)__builtin_amdgcn_ldexpf(xv1[i], sh);
    }
    const int dst = b >> 5;
    const int drow = (b & 31) * 8 + r;
    const int g = 8 * wave + (lane & 7);
    char* dws = p.ws[dst];
    const __amdgpu_buffer_rsrc_t rs = __builtin_amdgcn_make_buffer_rsrc(dws, 0, (int)WS_USED, 0x00020000);
    const unsigned off = A_OFF + par * A_PAR_BYTES + (unsigned)rank * A_SRC_BYTES + ((unsigned)g * 256u + (unsigned)drow) * 16u;
    __builtin_amdgcn_raw_buffer_store_b128(__builtin_bit_cast(u32x4, h), rs, (int)off, 0, AUX_SYS);
    asm volatile("s_waitcnt vmcnt(0)" ::: "memory");
    unsigned last = 0;
    if (lane == 0) last = __hip_atomic_fetch_add(s_cnt, 1u, __ATOMIC_RELAXED, __HIP_MEMORY_SCOPE_WORKGROUP);
    last = __builtin_amdgcn_readfirstlane(last);
    if (last == 3u) {
      const int rr = lane & 7;
      const float mm = fmaxf(fmaxf(s_rowmax[rr], s_rowmax[8 + rr]), fmaxf(s_rowmax[16 + rr], s_rowmax[24 + rr]));
      int ee = max(__builtin_amdgcn_frexp_expf(mm), -127) - 1;
      unsigned byte = ((unsigned)ee & 0xffu) << (8 * (rr & 3));
      byte |= __shfl_xor(byte, 1, 64);
      byte |= __shfl_xor(byte, 2, 64);
      const unsigned hi = __shfl(byte, 4, 64);
      if (lane == 0) {
        u32x4 f;
        f[0] = call; f[1] = byte; f[2] = call; f[3] = hi;
        const unsigned foff = FLAG_OFF + par * FLAG_PAR_BYTES + (unsigned)rank * FLAG_SRC_BYTES + (unsigned)(b & 31) * 16u;
        __builtin_amdgcn_raw_buffer_store_b128(f, rs, (int)foff, 0, AUX_SYS);
      }
    }
  }

  char* const wl = lds + wave * LDS_WAVE_BYTES;
  int esub[4];
  {
    int erun = -1000;
#pragma unroll
    for (int bt = 0; bt < 4; ++bt) {
      float m = 0.f;
#pragma unroll
      for (int j = 0; j < 8; ++j) m = fmaxf(m, absmax4(wv[bt][j]));
      m = wave_max(m);
      const int e = __builtin_amdgcn_readfirstlane(__builtin_amdgcn_frexp_expf(m));
      erun = max(erun, e);
      esub[bt] = erun;
      const int sh = 15 - erun;
#pragma unroll
      for (int i = 0; i < 4; ++i) {
        half8 h;
#pragma unroll
        for (int j = 0; j < 8; ++j) h[j] = (_Float16)__builtin_amdgcn_ldexpf(wv[bt][j][i], sh);
        *reinterpret_cast<half8*>(wl + ((8 * bt + (lane >> 3)) * 32 + 4 * (lane & 7) + i) * 16) = h;
      }
    }
  }
  __builtin_amdgcn_s_waitcnt(0xC07F);
  __builtin_amdgcn_wave_barrier();
  half8 wf[16];
#pragma unroll
  for (int s = 0; s < 16; ++s)
    wf[s] = *reinterpret_cast<const half8*>(wl + ((2 * s + (lane >> 5)) * 32 + (lane & 31)) * 16);

  const __amdgpu_buffer_rsrc_t rme = __builtin_amdgcn_make_buffer_rsrc(myws, 0, (int)WS_USED, 0x00020000);
  unsigned* const tmo = reinterpret_cast<unsigned*>(myws + TMO_OFF);
  u32x4 fl;
  {
    const unsigned foff = FLAG_OFF + par * FLAG_PAR_BYTES + (unsigned)wave * FLAG_SRC_BYTES + (unsigned)(m0 / 8 + (lane & 7)) * 16u;
    const unsigned long long t0 = __builtin_amdgcn_s_memrealtime();
    for (;;) {
      asm volatile("" ::: "memory");
      fl = __builtin_amdgcn_raw_buffer_load_b128(rme, (int)foff, 0, AUX_SYS);
      const bool ok = (fl[0] == call) && (fl[2] == call);
      if (__all(ok)) break;
      if (__hip_atomic_load(tmo, __ATOMIC_RELAXED, __HIP_MEMORY_SCOPE_AGENT) == call ||
          __builtin_amdgcn_s_memrealtime() - t0 > DEADLINE_TICKS) {
        __hip_atomic_store(tmo, call, __ATOMIC_RELAXED, __HIP_MEMORY_SCOPE_AGENT);
        break;
      }
      __builtin_amdgcn_s_sleep(1);
    }
    asm volatile("" ::: "memory");
  }

  half8 xf[2][16];
  {
    const unsigned base = A_OFF + par * A_PAR_BYTES + (unsigned)wave * A_SRC_BYTES +
                          ((unsigned)(lane >> 5) * 256u + (unsigned)(m0 + (lane & 31))) * 16u;
#pragma unroll
    for (int s = 0; s < 16; ++s)
#pragma unroll
      for (int mb = 0; mb < 2; ++mb)
        xf[mb][s] = __builtin_bit_cast(half8, __builtin_amdgcn_raw_buffer_load_b128(rme, (int)(base + (unsigned)s * 8192u + (unsigned)mb * 512u), 0, AUX_SYS));
  }
  int ex[2];
#pragma unroll
  for (int mb = 0; mb < 2; ++mb) {
    const int row = 32 * mb + (lane & 31);
    const unsigned lo = __shfl(fl[1], row >> 3, 64);
    const unsigned hi = __shfl(fl[3], row >> 3, 64);
    const unsigned wd = (row & 4) ? hi : lo;
    ex[mb] = (int)(signed char)((wd >> (8 * (row & 3))) & 0xffu) + 1;
  }

  f32x16 acc[2];
#pragma unroll
  for (int i = 0; i < 16; ++i) { acc[0][i] = 0.f; acc[1][i] = 0.f; }
#pragma unroll
  for (int bt = 0; bt < 4; ++bt) {
    if (bt > 0 && esub[bt] != esub[bt - 1]) {
      const int d = esub[bt - 1] - esub[bt];
#pragma unroll
      for (int i = 0; i < 16; ++i) {
        acc[0][i] = __builtin_amdgcn_ldexpf(acc[0][i], d);
        acc[1][i] = __builtin_amdgcn_ldexpf(acc[1][i], d);
      }
    }
#pragma unroll
    for (int s = 4 * bt; s < 4 * bt + 4; ++s) {
      acc[0] = __builtin_amdgcn_mfma_f32_32x32x16_f16(wf[s], xf[0][s], acc[0], 0, 0, 0);
      acc[1] = __builtin_amdgcn_mfma_f32_32x32x16_f16(wf[s], xf[1][s], acc[1], 0, 0, 0);
    }
  }

  {
    float* red = reinterpret_cast<float*>(wl);
#pragma unroll
    for (int mb = 0; mb < 2; ++mb) {
      const int sc = ex[mb] + esub[3] - 30;
#pragma unroll
      for (int q = 0; q < 4; ++q) {
        f32x4 v;
#pragma unroll
        for (int i = 0; i < 4; ++i) v[i] = __builtin_amdgcn_ldexpf(acc[mb][4 * q + i], sc);
        *reinterpret_cast<f32x4*>(red + (32 * mb + (lane & 31)) * RED_STRIDE + 8 * q + 4 * (lane >> 5)) = v;
      }
    }
  }
  __syncthreads();
  {
    const int row = tid >> 3, c4 = tid & 7;
    f32x4 sum = {0.f, 0.f, 0.f, 0.f};
#pragma unroll
    for (int wv2 = 0; wv2 < 8; ++wv2)
      sum += *reinterpret_cast<const f32x4*>(lds + wv2 * LDS_WAVE_BYTES + (row * RED_STRIDE + 4 * c4) * 4);
    const bool bad = __hip_atomic_load(tmo, __ATOMIC_RELAXED, __HIP_MEMORY_SCOPE_AGENT) == call;
    f32x4 o;
#pragma unroll
    for (int i = 0; i < 4; ++i) {
      const float y = sum[i];
      const float c = fminf(fmaxf(y, -60.f), 60.f);
      o[i] = y / (1.f + __expf(-c));
      if (bad) o[i] = __builtin_nanf("");
    }
    *reinterpret_cast<f32x4*>(p.out + (size_t)(m0 + row) * NDIM + n0 + 4 * c4) = o;
  }
}

}

extern "C" void kernel_launch(int rank, int nranks, ncclComm_t comm,
                              hipStream_t stream,
                              void* const* d_in, const int* in_sizes, int n_in,
                              void* d_out, int out_size,
                              const int* in_dtypes, int out_dtype,
                              void* const* d_out_all,
                              void* const* d_ws_all, size_t ws_bytes) {
  (void)comm; (void)in_sizes; (void)n_in; (void)out_size; (void)in_dtypes; (void)out_dtype; (void)d_out_all;
  if (nranks != NRANKS || (unsigned)rank >= (unsigned)NRANKS || ws_bytes < WS_USED) {
    fprintf(stderr, "kernel_launch: unexpected configuration\n");
    return;
  }
  Params p;
  p.x = static_cast<const float*>(d_in[0]);
  p.w = static_cast<const float*>(d_in[1]);
  p.out = static_cast<float*>(d_out);
  for (int i = 0; i < NRANKS; ++i) p.ws[i] = static_cast<char*>(d_ws_all[i]);
  p.rank = rank;
  a2a_gemm_silu<<<dim3(NWG), dim3(NTHREADS), 0, stream>>>(p);
  HIP_CHECK(hipGetLastError());
}
```

```cpp
#include <hip/hip_runtime.h>
#include <rccl/rccl.h>
#include <cstdio>
#include <cstdint>

#define HIP_CHECK(expr) do { hipError_t _e = (expr); if (_e != hipSuccess) { \
    fprintf(stderr, "HIP error %s at %s:%d\n", hipGetErrorString(_e), __FILE__, __LINE__); \
    return; } } while (0)

typedef _Float16 half8 __attribute__((ext_vector_type(8)));
typedef float f32x16 __attribute__((ext_vector_type(16)));
typedef float f32x4 __attribute__((ext_vector_type(4)));
typedef unsigned u32x4 __attribute__((ext_vector_type(4)));

namespace {

constexpr int NRANKS = 8;
constexpr int KBLK = 256;
constexpr int NDIM = 2048;
constexpr int NWG = 256;
constexpr int NTHREADS = 512;

constexpr unsigned A_SRC_BYTES = 32u * 256u * 16u;
constexpr unsigned A_PAR_BYTES = NRANKS * A_SRC_BYTES;
constexpr unsigned A_OFF = 0;
constexpr unsigned FLAG_OFF = 2u * A_PAR_BYTES;
constexpr unsigned FLAG_SRC_BYTES = 32u * 16u;
constexpr unsigned FLAG_PAR_BYTES = NRANKS * FLAG_SRC_BYTES;
constexpr unsigned CTR_OFF = FLAG_OFF + 2u * FLAG_PAR_BYTES;
constexpr unsigned TMO_OFF = CTR_OFF + NWG * 64u;
constexpr unsigned WS_USED = TMO_OFF + 128u;

#define STAMP(slot) do { } while (0)

constexpr unsigned long long DEADLINE_TICKS = 50000000ull;

constexpr int AUX_SYS = 17;

constexpr int LDS_WAVE_BYTES = 16384;
constexpr int RED_STRIDE = 36;
constexpr int LDS_MISC_OFF = 8 * LDS_WAVE_BYTES;
constexpr int LDS_BYTES = LDS_MISC_OFF + 256;

struct Params {
  const float* x;
  const float* w;
  float* out;
  char* ws[NRANKS];
  int rank;
};

__device__ __forceinline__ float wave_max(float v) {
  int x = __builtin_bit_cast(int, v);
  x = max(x, __builtin_amdgcn_update_dpp(0, x, 0xB1, 0xf, 0xf, false));
  x = max(x, __builtin_amdgcn_update_dpp(0, x, 0x4E, 0xf, 0xf, false));
  x = max(x, __builtin_amdgcn_update_dpp(0, x, 0x141, 0xf, 0xf, false));
  x = max(x, __builtin_amdgcn_update_dpp(0, x, 0x140, 0xf, 0xf, false));
  x = max(x, __builtin_amdgcn_update_dpp(0, x, 0x142, 0xa, 0xf, false));
  x = max(x, __builtin_amdgcn_update_dpp(0, x, 0x143, 0xc, 0xf, false));
  return __builtin_bit_cast(float, __builtin_amdgcn_readlane(x, 63));
}

__device__ __forceinline__ float absmax4(const f32x4 v) {
  return fmaxf(fmaxf(fabsf(v[0]), fabsf(v[1])), fmaxf(fabsf(v[2]), fabsf(v[3])));
}

__device__ __forceinline__ int stage_batch(const f32x4 (&wv)[8], int erun, char* wl, int slot, int lane) {
  float m = 0.f;
#pragma unroll
  for (int j = 0; j < 8; ++j) m = fmaxf(m, absmax4(wv[j]));
  m = wave_max(m);
  const int e = __builtin_amdgcn_readfirstlane(__builtin_amdgcn_frexp_expf(m));
  erun = max(erun, e);
  const int sh = 15 - erun;
#pragma unroll
  for (int i = 0; i < 4; ++i) {
    half8 h;
#pragma unroll
    for (int j = 0; j < 8; ++j) h[j] = (_Float16)__builtin_amdgcn_ldexpf(wv[j][i], sh);
    *reinterpret_cast<half8*>(wl + ((8 * slot + (lane >> 3)) * 32 + 4 * (lane & 7) + i) * 16) = h;
  }
  return erun;
}

__global__ void __launch_bounds__(NTHREADS, 2) a2a_gemm_silu(Params p) {
  __shared__ __attribute__((aligned(16))) char lds[LDS_BYTES];

  const int tid = threadIdx.x;
  const int lane = tid & 63;
  const int wave = __builtin_amdgcn_readfirstlane(tid >> 6);
  const int b = blockIdx.x;
  const int rank = p.rank;
  char* const myws = p.ws[rank];

  float* const s_rowmax = reinterpret_cast<float*>(lds + LDS_MISC_OFF);
  unsigned* const s_call = reinterpret_cast<unsigned*>(lds + LDS_MISC_OFF + 128);
  unsigned* const s_cnt = reinterpret_cast<unsigned*>(lds + LDS_MISC_OFF + 132);
  unsigned* const s_bad = reinterpret_cast<unsigned*>(lds + LDS_MISC_OFF + 136);

  const int xcd = b & 7, idx = b >> 3;
  const int n0 = xcd * 256 + (idx & 7) * 32;
  const int mt = idx >> 3;
  const int m0 = mt * 64;
  const int rot = mt;

  if (wave == 0) STAMP(0);
  f32x4 wv[4][8];
  const float* wsrc = p.w + (size_t)(KBLK * wave + 8 * (lane >> 3)) * NDIM + n0 + 4 * (lane & 7);

  f32x4 xv0 = {0.f, 0.f, 0.f, 0.f}, xv1 = {0.f, 0.f, 0.f, 0.f};
  if (wave < 4) {
    const float* src = p.x + (size_t)(8 * b + (lane >> 3)) * KBLK + 64 * wave + 8 * (lane & 7);
    xv0 = *reinterpret_cast<const f32x4*>(src);
    xv1 = *reinterpret_cast<const f32x4*>(src + 4);
    int m = __builtin_bit_cast(int, fmaxf(absmax4(xv0), absmax4(xv1)));
    m = max(m, __builtin_amdgcn_update_dpp(0, m, 0xB1, 0xf, 0xf, false));
    m = max(m, __builtin_amdgcn_update_dpp(0, m, 0x4E, 0xf, 0xf, false));
    m = max(m, __builtin_amdgcn_update_dpp(0, m, 0x141, 0xf, 0xf, false));
    if ((lane & 7) == 0) s_rowmax[wave * 8 + (lane >> 3)] = __builtin_bit_cast(float, m);
  } else {
    if (tid == NTHREADS - 64) {
      unsigned* ctr = reinterpret_cast<unsigned*>(myws + CTR_OFF + 64u * (unsigned)b);
      const unsigned t = __hip_atomic_fetch_add(ctr, 1u, __ATOMIC_RELAXED, __HIP_MEMORY_SCOPE_AGENT);
      *s_call = t + 1u;
      *s_cnt = 0u;
      *s_bad = 0u;
    }
  }
  __syncthreads();
  const unsigned call = *s_call;
  const unsigned par = call & 1u;

  if (wave < 4) {
    const int r = lane >> 3;
    const float m = fmaxf(fmaxf(s_rowmax[r], s_rowmax[8 + r]), fmaxf(s_rowmax[16 + r], s_rowmax[24 + r]));
    int e = __builtin_amdgcn_frexp_expf(m);
    e = max(e, -127);
    const int sh = 15 - e;
    half8 h;
#pragma unroll
    for (int i = 0; i < 4; ++i) {
      h[i] = (_Float16)__builtin_amdgcn_ldexpf(xv0[i], sh);
      h[4 + i] = (_Float16)__builtin_amdgcn_ldexpf(xv1[i], sh);
    }
    const int dst = b >> 5;
    const int drow = (b & 31) * 8 + r;
    const int g = 8 * wave + (lane & 7);
    char* dws = p.ws[dst];
    const __amdgpu_buffer_rsrc_t rs = __builtin_amdgcn_make_buffer_rsrc(dws, 0, (int)WS_USED, 0x00020000);
    const unsigned off = A_OFF + par * A_PAR_BYTES + (unsigned)rank * A_SRC_BYTES + ((unsigned)g * 256u + (unsigned)drow) * 16u;
    __builtin_amdgcn_raw_buffer_store_b128(__builtin_bit_cast(u32x4, h), rs, (int)off, 0, AUX_SYS);
    __builtin_amdgcn_sched_barrier(0);
#pragma unroll
    for (int r2 = 0; r2 < 4; ++r2)
#pragma unroll
      for (int j = 0; j < 8; ++j)
        wv[r2][j] = *reinterpret_cast<const f32x4*>(wsrc + (size_t)(64 * ((rot + r2) & 3) + j) * NDIM);
    __builtin_amdgcn_sched_barrier(0);
    asm volatile("s_waitcnt vmcnt(32)" ::: "memory");
    if (wave == 0) STAMP(3);
    unsigned last = 0;
    if (lane == 0) last = __hip_atomic_fetch_add(s_cnt, 1u, __ATOMIC_RELAXED, __HIP_MEMORY_SCOPE_WORKGROUP);
    last = __builtin_amdgcn_readfirstlane(last);
    if (last == 3u) {
      const int rr = lane & 7;
      const float mm = fmaxf(fmaxf(s_rowmax[rr], s_rowmax[8 + rr]), fmaxf(s_rowmax[16 + rr], s_rowmax[24 + rr]));
      int ee = max(__builtin_amdgcn_frexp_expf(mm), -127) - 1;
      unsigned byte = ((unsigned)ee & 0xffu) << (8 * (rr & 3));
      byte |= __shfl_xor(byte, 1, 64);
      byte |= __shfl_xor(byte, 2, 64);
      const unsigned hi = __shfl(byte, 4, 64);
      if (lane == 0) {
        u32x4 f;
        f[0] = call; f[1] = byte; f[2] = call; f[3] = hi;
        const unsigned foff = FLAG_OFF + par * FLAG_PAR_BYTES + (unsigned)rank * FLAG_SRC_BYTES + (unsigned)(b & 31) * 16u;
        __builtin_amdgcn_raw_buffer_store_b128(f, rs, (int)foff, 0, AUX_SYS);
      }
    }
  } else {
#pragma unroll
    for (int r2 = 0; r2 < 4; ++r2)
#pragma unroll
      for (int j = 0; j < 8; ++j)
        wv[r2][j] = *reinterpret_cast<const f32x4*>(wsrc + (size_t)(64 * ((rot + r2) & 3) + j) * NDIM);
  }

  char* const wl = lds + wave * LDS_WAVE_BYTES;
  int esub[4];
  esub[0] = stage_batch(wv[0], -1000, wl, 0, lane);
  esub[1] = stage_batch(wv[1], esub[0], wl, 1, lane);
  esub[2] = stage_batch(wv[2], esub[1], wl, 2, lane);
  esub[3] = stage_batch(wv[3], esub[2], wl, 3, lane);
  half8 wf[16];
#pragma unroll
  for (int s = 0; s < 16; ++s)
    wf[s] = *reinterpret_cast<const half8*>(wl + ((2 * s + (lane >> 5)) * 32 + (lane & 31)) * 16);
  if (wave == 2) STAMP(1);
  if (wave == 6) STAMP(2);
  __builtin_amdgcn_sched_barrier(0);

  const __amdgpu_buffer_rsrc_t rme = __builtin_amdgcn_make_buffer_rsrc(myws, 0, (int)WS_USED, 0x00020000);
  u32x4 fl;
  {
    unsigned* const tmo = reinterpret_cast<unsigned*>(myws + TMO_OFF);
    const unsigned foff = FLAG_OFF + par * FLAG_PAR_BYTES + (unsigned)wave * FLAG_SRC_BYTES + (unsigned)(m0 / 8 + (lane & 7)) * 16u;
    const unsigned long long t0 = __builtin_amdgcn_s_memrealtime();
    for (unsigned it = 1;; ++it) {
      asm volatile("" ::: "memory");
      fl = __builtin_amdgcn_raw_buffer_load_b128(rme, (int)foff, 0, AUX_SYS);
      const bool ok = (fl[0] == call) && (fl[2] == call);
      if (__all(ok)) break;
      if ((it & 15u) == 0u) {
        if (__hip_atomic_load(tmo, __ATOMIC_RELAXED, __HIP_MEMORY_SCOPE_AGENT) == call ||
            __builtin_amdgcn_s_memrealtime() - t0 > DEADLINE_TICKS) {
          __hip_atomic_store(tmo, call, __ATOMIC_RELAXED, __HIP_MEMORY_SCOPE_AGENT);
          *s_bad = 1u;
          break;
        }
      }
    }
    asm volatile("" ::: "memory");
  }
  if (wave == 2) STAMP(4);
  if (wave == 6) STAMP(8);

  half8 xf[2][16];
  {
    const unsigned base = A_OFF + par * A_PAR_BYTES + (unsigned)wave * A_SRC_BYTES +
                          ((unsigned)(lane >> 5) * 256u + (unsigned)(m0 + (lane & 31))) * 16u;
#pragma unroll
    for (int r2 = 0; r2 < 4; ++r2) {
      const unsigned bb = base + (unsigned)((rot + r2) & 3) * 32768u;
#pragma unroll
      for (int s = 0; s < 4; ++s)
#pragma unroll
        for (int mb = 0; mb < 2; ++mb)
          xf[mb][4 * r2 + s] = __builtin_bit_cast(half8, __builtin_amdgcn_raw_buffer_load_b128(rme, (int)(bb + (unsigned)s * 8192u + (unsigned)mb * 512u), 0, AUX_SYS));
    }
  }
  __builtin_amdgcn_sched_barrier(0);

  int ex[2];
#pragma unroll
  for (int mb = 0; mb < 2; ++mb) {
    const int row = 32 * mb + (lane & 31);
    const unsigned lo = __shfl(fl[1], row >> 3, 64);
    const unsigned hi = __shfl(fl[3], row >> 3, 64);
    const unsigned wd = (row & 4) ? hi : lo;
    ex[mb] = (int)(signed char)((wd >> (8 * (row & 3))) & 0xffu) + 1;
  }

  f32x16 acc[2];
#pragma unroll
  for (int i = 0; i < 16; ++i) { acc[0][i] = 0.f; acc[1][i] = 0.f; }

#pragma unroll
  for (int r2 = 0; r2 < 4; ++r2) {
    if (r2 > 0 && esub[r2] != esub[r2 - 1]) {
      const int d = esub[r2 - 1] - esub[r2];
#pragma unroll
      for (int i = 0; i < 16; ++i) {
        acc[0][i] = __builtin_amdgcn_ldexpf(acc[0][i], d);
        acc[1][i] = __builtin_amdgcn_ldexpf(acc[1][i], d);
      }
    }
#pragma unroll
    for (int s = 0; s < 4; ++s) {
      acc[0] = __builtin_amdgcn_mfma_f32_32x32x16_f16(wf[4 * r2 + s], xf[0][4 * r2 + s], acc[0], 0, 0, 0);
      acc[1] = __builtin_amdgcn_mfma_f32_32x32x16_f16(wf[4 * r2 + s], xf[1][4 * r2 + s], acc[1], 0, 0, 0);
    }
  }

  {
    float* red = reinterpret_cast<float*>(wl);
#pragma unroll
    for (int mb = 0; mb < 2; ++mb) {
      const int sc = ex[mb] + esub[3] - 30;
#pragma unroll
      for (int q = 0; q < 4; ++q) {
        f32x4 v;
#pragma unroll
        for (int i = 0; i < 4; ++i) v[i] = __builtin_amdgcn_ldexpf(acc[mb][4 * q + i], sc);
        *reinterpret_cast<f32x4*>(red + (32 * mb + (lane & 31)) * RED_STRIDE + 8 * q + 4 * (lane >> 5)) = v;
      }
    }
  }
  __syncthreads();
  if (wave == 0) STAMP(14);
  {
    const int row = tid >> 3, c4 = tid & 7;
    f32x4 sum = {0.f, 0.f, 0.f, 0.f};
#pragma unroll
    for (int wv2 = 0; wv2 < 8; ++wv2)
      sum += *reinterpret_cast<const f32x4*>(lds + wv2 * LDS_WAVE_BYTES + (row * RED_STRIDE + 4 * c4) * 4);
    const bool bad = *s_bad != 0u;
    f32x4 o;
#pragma unroll
    for (int i = 0; i < 4; ++i) {
      const float y = sum[i];
      const float c = fminf(fmaxf(y, -60.f), 60.f);
      o[i] = y / (1.f + __expf(-c));
      if (bad) o[i] = __builtin_nanf("");
    }
    *reinterpret_cast<f32x4*>(p.out + (size_t)(m0 + row) * NDIM + n0 + 4 * c4) = o;
  }
}

}

extern "C" void kernel_launch(int rank, int nranks, ncclComm_t comm,
                              hipStream_t stream,
                              void* const* d_in, const int* in_sizes, int n_in,
                              void* d_out, int out_size,
                              const int* in_dtypes, int out_dtype,
                              void* const* d_out_all,
                              void* const* d_ws_all, size_t ws_bytes) {
  (void)comm; (void)in_sizes; (void)n_in; (void)out_size; (void)in_dtypes; (void)out_dtype; (void)d_out_all;
  if (nranks != NRANKS || (unsigned)rank >= (unsigned)NRANKS || ws_bytes < WS_USED) {
    fprintf(stderr, "kernel_launch: unexpected configuration\n");
    return;
  }
  Params p;
  p.x = static_cast<const float*>(d_in[0]);
  p.w = static_cast<const float*>(d_in[1]);
  p.out = static_cast<float*>(d_out);
  for (int i = 0; i < NRANKS; ++i) p.ws[i] = static_cast<char*>(d_ws_all[i]);
  p.rank = rank;
  a2a_gemm_silu<<<dim3(NWG), dim3(NTHREADS), 0, stream>>>(p);
  HIP_CHECK(hipGetLastError());
}
```

```cpp
#include <hip/hip_runtime.h>
#include <rccl/rccl.h>
#include <cstdio>
#include <cstdint>

#define HIP_CHECK(expr) do { hipError_t _e = (expr); if (_e != hipSuccess) { \
    fprintf(stderr, "HIP error %s at %s:%d\n", hipGetErrorString(_e), __FILE__, __LINE__); \
    return; } } while (0)

typedef _Float16 half8 __attribute__((ext_vector_type(8)));
typedef float f32x16 __attribute__((ext_vector_type(16)));
typedef float f32x4 __attribute__((ext_vector_type(4)));
typedef unsigned u32x4 __attribute__((ext_vector_type(4)));

namespace {

constexpr int NRANKS = 8;
constexpr int KBLK = 256;
constexpr int NDIM = 2048;
constexpr int NWG = 256;
constexpr int NTHREADS = 512;

constexpr unsigned A_SRC_BYTES = 32u * 256u * 16u;
constexpr unsigned A_PAR_BYTES = NRANKS * A_SRC_BYTES;
constexpr unsigned A_OFF = 0;
constexpr unsigned FLAG_OFF = 2u * A_PAR_BYTES;
constexpr unsigned FLAG_SRC_BYTES = 32u * 16u;
constexpr unsigned FLAG_PAR_BYTES = NRANKS * FLAG_SRC_BYTES;
constexpr unsigned CTR_OFF = FLAG_OFF + 2u * FLAG_PAR_BYTES;
constexpr unsigned TMO_OFF = CTR_OFF + NWG * 64u;
constexpr unsigned WS_USED = TMO_OFF + 128u;

constexpr unsigned long long DEADLINE_TICKS = 50000000ull;

constexpr int AUX_SYS = 17;

constexpr int LDS_WAVE_BYTES = 16384;
constexpr int RED_STRIDE = 36;
constexpr int LDS_MISC_OFF = 8 * LDS_WAVE_BYTES;
constexpr int LDS_BYTES = LDS_MISC_OFF + 256;

struct Params {
  const float* x;
  const float* w;
  float* out;
  char* ws[NRANKS];
  int rank;
};

__device__ __forceinline__ float wave_max(float v) {
  int x = __builtin_bit_cast(int, v);
  x = max(x, __builtin_amdgcn_update_dpp(0, x, 0xB1, 0xf, 0xf, false));
  x = max(x, __builtin_amdgcn_update_dpp(0, x, 0x4E, 0xf, 0xf, false));
  x = max(x, __builtin_amdgcn_update_dpp(0, x, 0x141, 0xf, 0xf, false));
  x = max(x, __builtin_amdgcn_update_dpp(0, x, 0x140, 0xf, 0xf, false));
  x = max(x, __builtin_amdgcn_update_dpp(0, x, 0x142, 0xa, 0xf, false));
  x = max(x, __builtin_amdgcn_update_dpp(0, x, 0x143, 0xc, 0xf, false));
  return __builtin_bit_cast(float, __builtin_amdgcn_readlane(x, 63));
}

__device__ __forceinline__ float absmax4(const f32x4 v) {
  return fmaxf(fmaxf(fabsf(v[0]), fabsf(v[1])), fmaxf(fabsf(v[2]), fabsf(v[3])));
}

__device__ __forceinline__ int stage_batch(const f32x4 (&wv)[8], int erun, char* wl, int slot, int lane) {
  float m = 0.f;
#pragma unroll
  for (int j = 0; j < 8; ++j) m = fmaxf(m, absmax4(wv[j]));
  m = wave_max(m);
  const int e = __builtin_amdgcn_readfirstlane((m > 0.f) ? __builtin_amdgcn_frexp_expf(m) : -1000);
  erun = max(erun, e);
  const int sh = 15 - erun;
#pragma unroll
  for (int i = 0; i < 4; ++i) {
    half8 h;
#pragma unroll
    for (int j = 0; j < 8; ++j) h[j] = (_Float16)__builtin_amdgcn_ldexpf(wv[j][i], sh);
    *reinterpret_cast<half8*>(wl + ((8 * slot + (lane >> 3)) * 32 + 4 * (lane & 7) + i) * 16) = h;
  }
  return erun;
}

__global__ void __launch_bounds__(NTHREADS, 2) a2a_gemm_silu(Params p) {
  __shared__ __attribute__((aligned(16))) char lds[LDS_BYTES];

  const int tid = threadIdx.x;
  const int lane = tid & 63;
  const int wave = __builtin_amdgcn_readfirstlane(tid >> 6);
  const int b = blockIdx.x;
  const int rank = p.rank;
  char* const myws = p.ws[rank];

  float* const s_rowmax = reinterpret_cast<float*>(lds + LDS_MISC_OFF);
  unsigned* const s_call = reinterpret_cast<unsigned*>(lds + LDS_MISC_OFF + 128);
  unsigned* const s_cnt = reinterpret_cast<unsigned*>(lds + LDS_MISC_OFF + 132);
  unsigned* const s_bad = reinterpret_cast<unsigned*>(lds + LDS_MISC_OFF + 136);

  const int xcd = b & 7, idx = b >> 3;
  const int n0 = xcd * 256 + (idx & 7) * 32;
  const int mt = idx >> 3;
  const int m0 = mt * 64;
  const int rot = mt;

  f32x4 wv[4][8];
  const float* wsrc = p.w + (size_t)(KBLK * wave + 8 * (lane >> 3)) * NDIM + n0 + 4 * (lane & 7);

  f32x4 xv0 = {0.f, 0.f, 0.f, 0.f}, xv1 = {0.f, 0.f, 0.f, 0.f};
  if (wave < 4) {
    const float* src = p.x + (size_t)(8 * b + (lane >> 3)) * KBLK + 64 * wave + 8 * (lane & 7);
    xv0 = *reinterpret_cast<const f32x4*>(src);
    xv1 = *reinterpret_cast<const f32x4*>(src + 4);
    int m = __builtin_bit_cast(int, fmaxf(absmax4(xv0), absmax4(xv1)));
    m = max(m, __builtin_amdgcn_update_dpp(0, m, 0xB1, 0xf, 0xf, false));
    m = max(m, __builtin_amdgcn_update_dpp(0, m, 0x4E, 0xf, 0xf, false));
    m = max(m, __builtin_amdgcn_update_dpp(0, m, 0x141, 0xf, 0xf, false));
    if ((lane & 7) == 0) s_rowmax[wave * 8 + (lane >> 3)] = __builtin_bit_cast(float, m);
  } else {
    if (tid == NTHREADS - 64) {
      unsigned* ctr = reinterpret_cast<unsigned*>(myws + CTR_OFF + 64u * (unsigned)b);
      const unsigned t = __hip_atomic_fetch_add(ctr, 1u, __ATOMIC_RELAXED, __HIP_MEMORY_SCOPE_AGENT);
      *s_call = t + 1u;
      *s_cnt = 0u;
      *s_bad = 0u;
    }
#pragma unroll
    for (int j = 0; j < 8; ++j)
      wv[0][j] = *reinterpret_cast<const f32x4*>(wsrc + (size_t)(64 * rot + j) * NDIM);
  }
  __syncthreads();
  const unsigned call = *s_call;
  const unsigned par = call & 1u;

  if (wave < 4) {
    const int r = lane >> 3;
    const float m = fmaxf(fmaxf(s_rowmax[r], s_rowmax[8 + r]), fmaxf(s_rowmax[16 + r], s_rowmax[24 + r]));
    int e = __builtin_amdgcn_frexp_expf(m);
    e = max(e, -127);
    const int sh = 15 - e;
    half8 h;
#pragma unroll
    for (int i = 0; i < 4; ++i) {
      h[i] = (_Float16)__builtin_amdgcn_ldexpf(xv0[i], sh);
      h[4 + i] = (_Float16)__builtin_amdgcn_ldexpf(xv1[i], sh);
    }
    const int dst = b >> 5;
    const int drow = (b & 31) * 8 + r;
    const int g = 8 * wave + (lane & 7);
    char* dws = p.ws[dst];
    const __amdgpu_buffer_rsrc_t rs = __builtin_amdgcn_make_buffer_rsrc(dws, 0, (int)WS_USED, 0x00020000);
    const unsigned off = A_OFF + par * A_PAR_BYTES + (unsigned)rank * A_SRC_BYTES + ((unsigned)g * 256u + (unsigned)drow) * 16u;
    __builtin_amdgcn_raw_buffer_store_b128(__builtin_bit_cast(u32x4, h), rs, (int)off, 0, AUX_SYS);
    __builtin_amdgcn_sched_barrier(0);
#pragma unroll
    for (int j = 0; j < 8; ++j)
      wv[0][j] = *reinterpret_cast<const f32x4*>(wsrc + (size_t)(64 * rot + j) * NDIM);
    __builtin_amdgcn_sched_barrier(0);
    asm volatile("s_waitcnt vmcnt(8)" ::: "memory");
    unsigned last = 0;
    if (lane == 0) last = __hip_atomic_fetch_add(s_cnt, 1u, __ATOMIC_RELAXED, __HIP_MEMORY_SCOPE_WORKGROUP);
    last = __builtin_amdgcn_readfirstlane(last);
    if (last == 3u) {
      const int rr = lane & 7;
      const float mm = fmaxf(fmaxf(s_rowmax[rr], s_rowmax[8 + rr]), fmaxf(s_rowmax[16 + rr], s_rowmax[24 + rr]));
      int ee = max(__builtin_amdgcn_frexp_expf(mm), -127) - 1;
      unsigned byte = ((unsigned)ee & 0xffu) << (8 * (rr & 3));
      byte |= __shfl_xor(byte, 1, 64);
      byte |= __shfl_xor(byte, 2, 64);
      const unsigned hi = __shfl(byte, 4, 64);
      if (lane == 0) {
        u32x4 f;
        f[0] = call; f[1] = byte; f[2] = call; f[3] = hi;
        const unsigned foff = FLAG_OFF + par * FLAG_PAR_BYTES + (unsigned)rank * FLAG_SRC_BYTES + (unsigned)(b & 31) * 16u;
        __builtin_amdgcn_raw_buffer_store_b128(f, rs, (int)foff, 0, AUX_SYS);
      }
    }
  }

  char* const wl = lds + wave * LDS_WAVE_BYTES;
  int esub[4];
  esub[0] = stage_batch(wv[0], -1000, wl, 0, lane);
  __builtin_amdgcn_sched_barrier(0);
#pragma unroll
  for (int r2 = 1; r2 < 4; ++r2)
#pragma unroll
    for (int j = 0; j < 8; ++j)
      wv[r2][j] = *reinterpret_cast<const f32x4*>(wsrc + (size_t)(64 * ((rot + r2) & 3) + j) * NDIM);
  __builtin_amdgcn_sched_barrier(0);
  esub[1] = stage_batch(wv[1], esub[0], wl, 1, lane);
  __builtin_amdgcn_sched_barrier(0);
  const __amdgpu_buffer_rsrc_t rme = __builtin_amdgcn_make_buffer_rsrc(myws, 0, (int)WS_USED, 0x00020000);
  const unsigned foff = FLAG_OFF + par * FLAG_PAR_BYTES + (unsigned)wave * FLAG_SRC_BYTES + (unsigned)(m0 / 8 + (lane & 7)) * 16u;
  asm volatile("" ::: "memory");
  u32x4 fl = __builtin_amdgcn_raw_buffer_load_b128(rme, (int)foff, 0, AUX_SYS);
  asm volatile("" ::: "memory");
  __builtin_amdgcn_sched_barrier(0);
  esub[2] = stage_batch(wv[2], esub[1], wl, 2, lane);
  esub[3] = stage_batch(wv[3], esub[2], wl, 3, lane);
  half8 wf[16];
#pragma unroll
  for (int s = 0; s < 16; ++s)
    wf[s] = *reinterpret_cast<const half8*>(wl + ((2 * s + (lane >> 5)) * 32 + (lane & 31)) * 16);
  __builtin_amdgcn_sched_barrier(0);

  if (!__all((fl[0] == call) && (fl[2] == call))) {
    unsigned* const tmo = reinterpret_cast<unsigned*>(myws + TMO_OFF);
    const unsigned long long t0 = __builtin_amdgcn_s_memrealtime();
    for (unsigned it = 1;; ++it) {
      asm volatile("" ::: "memory");
      fl = __builtin_amdgcn_raw_buffer_load_b128(rme, (int)foff, 0, AUX_SYS);
      const bool ok = (fl[0] == call) && (fl[2] == call);
      if (__all(ok)) break;
      if ((it & 15u) == 0u) {
        if (__hip_atomic_load(tmo, __ATOMIC_RELAXED, __HIP_MEMORY_SCOPE_AGENT) == call ||
            __builtin_amdgcn_s_memrealtime() - t0 > DEADLINE_TICKS) {
          __hip_atomic_store(tmo, call, __ATOMIC_RELAXED, __HIP_MEMORY_SCOPE_AGENT);
          *s_bad = 1u;
          break;
        }
      }
    }
  }
  asm volatile("" ::: "memory");

  half8 xf[2][16];
  {
    const unsigned base = A_OFF + par * A_PAR_BYTES + (unsigned)wave * A_SRC_BYTES +
                          ((unsigned)(lane >> 5) * 256u + (unsigned)(m0 + (lane & 31))) * 16u;
#pragma unroll
    for (int r2 = 0; r2 < 4; ++r2) {
      const unsigned bb = base + (unsigned)((rot + r2) & 3) * 32768u;
#pragma unroll
      for (int s = 0; s < 4; ++s)
#pragma unroll
        for (int mb = 0; mb < 2; ++mb)
          xf[mb][4 * r2 + s] = __builtin_bit_cast(half8, __builtin_amdgcn_raw_buffer_load_b128(rme, (int)(bb + (unsigned)s * 8192u + (unsigned)mb * 512u), 0, AUX_SYS));
    }
  }
  __builtin_amdgcn_sched_barrier(0);

  int ex[2];
#pragma unroll
  for (int mb = 0; mb < 2; ++mb) {
    const int row = 32 * mb + (lane & 31);
    const unsigned lo = __shfl(fl[1], row >> 3, 64);
    const unsigned hi = __shfl(fl[3], row >> 3, 64);
    const unsigned wd = (row & 4) ? hi : lo;
    ex[mb] = (int)(signed char)((wd >> (8 * (row & 3))) & 0xffu) + 1;
  }

  f32x16 acc[2];
#pragma unroll
  for (int i = 0; i < 16; ++i) { acc[0][i] = 0.f; acc[1][i] = 0.f; }

#pragma unroll
  for (int r2 = 0; r2 < 4; ++r2) {
    if (r2 > 0 && esub[r2] != esub[r2 - 1]) {
      const int d = esub[r2 - 1] - esub[r2];
#pragma unroll
      for (int i = 0; i < 16; ++i) {
        acc[0][i] = __builtin_amdgcn_ldexpf(acc[0][i], d);
        acc[1][i] = __builtin_amdgcn_ldexpf(acc[1][i], d);
      }
    }
#pragma unroll
    for (int s = 0; s < 4; ++s) {
      acc[0] = __builtin_amdgcn_mfma_f32_32x32x16_f16(wf[4 * r2 + s], xf[0][4 * r2 + s], acc[0], 0, 0, 0);
      acc[1] = __builtin_amdgcn_mfma_f32_32x32x16_f16(wf[4 * r2 + s], xf[1][4 * r2 + s], acc[1], 0, 0, 0);
    }
  }

  {
    float* red = reinterpret_cast<float*>(wl);
#pragma unroll
    for (int mb = 0; mb < 2; ++mb) {
      const int sc = ex[mb] + esub[3] - 30;
#pragma unroll
      for (int q = 0; q < 4; ++q) {
        f32x4 v;
#pragma unroll
        for (int i = 0; i < 4; ++i) v[i] = __builtin_amdgcn_ldexpf(acc[mb][4 * q + i], sc);
        *reinterpret_cast<f32x4*>(red + (32 * mb + (lane & 31)) * RED_STRIDE + 8 * q + 4 * (lane >> 5)) = v;
      }
    }
  }
  __syncthreads();
  {
    const int row = tid >> 3, c4 = tid & 7;
    f32x4 sum = {0.f, 0.f, 0.f, 0.f};
#pragma unroll
    for (int wv2 = 0; wv2 < 8; ++wv2)
      sum += *reinterpret_cast<const f32x4*>(lds + wv2 * LDS_WAVE_BYTES + (row * RED_STRIDE + 4 * c4) * 4);
    const bool bad = *s_bad != 0u;
    f32x4 o;
#pragma unroll
    for (int i = 0; i < 4; ++i) {
      const float y = sum[i];
      const float c = fminf(fmaxf(y, -60.f), 60.f);
      o[i] = y / (1.f + __expf(-c));
      if (bad) o[i] = __builtin_nanf("");
    }
    *reinterpret_cast<f32x4*>(p.out + (size_t)(m0 + row) * NDIM + n0 + 4 * c4) = o;
  }
}

}

extern "C" void kernel_launch(int rank, int nranks, ncclComm_t comm,
                              hipStream_t stream,
                              void* const* d_in, const int* in_sizes, int n_in,
                              void* d_out, int out_size,
                              const int* in_dtypes, int out_dtype,
                              void* const* d_out_all,
                              void* const* d_ws_all, size_t ws_bytes) {
  (void)comm; (void)in_sizes; (void)n_in; (void)out_size; (void)in_dtypes; (void)out_dtype; (void)d_out_all;
  if (nranks != NRANKS || (unsigned)rank >= (unsigned)NRANKS || ws_bytes < WS_USED) {
    fprintf(stderr, "kernel_launch: unexpected configuration\n");
    return;
  }
  Params p;
  p.x = static_cast<const float*>(d_in[0]);
  p.w = static_cast<const float*>(d_in[1]);
  p.out = static_cast<float*>(d_out);
  for (int i = 0; i < NRANKS; ++i) p.ws[i] = static_cast<char*>(d_ws_all[i]);
  p.rank = rank;
  a2a_gemm_silu<<<dim3(NWG), dim3(NTHREADS), 0, stream>>>(p);
  HIP_CHECK(hipGetLastError());
}
```
